```python
import numpy as np

kernel_name = "gpu_gemm_rs_m4096_k4096_n4096_e4m3e4m3_dequant"


ABSMAX_RTOL = 0.03

COMPUTE_DTYPE = "float32"
EPILOGUE = "dequant"
IN_QUANT = None
OUT_QUANT = None
IN_FP8 = "e4m3e4m3"
IN_DTYPES = {"x": "e4m3", "w_mat": "e4m3",
             "scale_x": "float32", "scale_w": "float32"}
WORLD_SIZE = 8

def _snap_fp8(x, mant_bits, max_abs, min_normal):
    a = np.abs(x.astype(np.float64))
    _m, _e = np.frexp(a)
    q = np.where(a >= min_normal, np.ldexp(1.0, _e - 1 - mant_bits),
                 min_normal * 2.0 ** -mant_bits)
    snapped = np.minimum(np.round(a / q) * q, max_abs)
    return (np.sign(x) * snapped).astype(np.float32)

def setup_inputs():

    x = _snap_fp8(np.random.randn(4096, 4096).astype(np.float32), 3, 448.0, 2.0 ** -6)
    w_mat = _snap_fp8(np.random.randn(4096, 4096).astype(np.float32), 3, 448.0, 2.0 ** -6)
    scale_x = (0.02 * (1.0 + np.random.rand(1))).astype(np.float32)
    scale_w = (0.02 * (1.0 + np.random.rand(1))).astype(np.float32)
    return {"x": x, "w_mat": w_mat, "scale_x": scale_x, "scale_w": scale_w}

def reference(x, w_mat, scale_x, scale_w):
    acc = (x @ w_mat).astype(np.float32)
    y = (acc * (scale_x * scale_w)).astype(np.float32)
    return y
```

```cpp
#include <hip/hip_runtime.h>
#include <rccl/rccl.h>
#include <cstdio>
#include <cstdint>

#define HIP_CHECK(expr) do { hipError_t _e = (expr); if (_e != hipSuccess) { \
    fprintf(stderr, "HIP error %s at %s:%d\n", hipGetErrorString(_e), __FILE__, __LINE__); \
    return; } } while (0)

typedef int v8i __attribute__((ext_vector_type(8)));
typedef int v4i __attribute__((ext_vector_type(4)));
typedef float v16f __attribute__((ext_vector_type(16)));
typedef float v4f __attribute__((ext_vector_type(4)));

constexpr int NR = 8;
constexpr int GM = 4096, GN = 4096, GK = 4096;
constexpr int KS = GK / NR;
constexpr int MS = GM / NR;
constexpr int ROUNDS = KS / 64;
constexpr int NWG = 256;

constexpr size_t OFF_TMO = 0;
constexpr size_t OFF_CTR = 4096;
constexpr size_t OFF_FLAG = 65536;
constexpr size_t OFF_DATA = 1u << 20;
constexpr size_t W_BYTES = (size_t)NR * KS * GN;
constexpr size_t X_BYTES = (size_t)NR * KS * MS;
constexpr size_t BUF_BYTES = W_BYTES + X_BYTES;
constexpr size_t WS_NEEDED = OFF_DATA + 2 * BUF_BYTES;

constexpr unsigned long long DEADLINE_TICKS = 50000000ull;

#define AUX_SYS 17

__device__ __forceinline__ void tr4x4(unsigned d0, unsigned d1, unsigned d2, unsigned d3,
                                      unsigned& o0, unsigned& o1, unsigned& o2, unsigned& o3) {
    unsigned t0 = __builtin_amdgcn_perm(d1, d0, 0x05010400u);
    unsigned t1 = __builtin_amdgcn_perm(d1, d0, 0x07030602u);
    unsigned t2 = __builtin_amdgcn_perm(d3, d2, 0x05010400u);
    unsigned t3 = __builtin_amdgcn_perm(d3, d2, 0x07030602u);
    o0 = __builtin_amdgcn_perm(t2, t0, 0x05040100u);
    o1 = __builtin_amdgcn_perm(t2, t0, 0x07060302u);
    o2 = __builtin_amdgcn_perm(t3, t1, 0x05040100u);
    o3 = __builtin_amdgcn_perm(t3, t1, 0x07060302u);
}

__global__ void __launch_bounds__(256, 1)
gemm_ag_kernel(const uint8_t* __restrict__ x, const uint8_t* __restrict__ w,
               const float* __restrict__ sx, const float* __restrict__ sw,
               float* __restrict__ out, void* const* __restrict__ ws_all, int rank) {
    __shared__ unsigned s_call;
    const int b = blockIdx.x;
    const int tid = threadIdx.x;
    const int wave = __builtin_amdgcn_readfirstlane(tid >> 6);
    const int lane = tid & 63;
    const int jj = lane & 31, h = lane >> 5;

    char* ws_me = (char*)ws_all[rank];
    unsigned* tmo = (unsigned*)(ws_me + OFF_TMO);

    if (tid == 0) {
        unsigned* ctr = (unsigned*)(ws_me + OFF_CTR + (size_t)b * 64);
        unsigned c = __hip_atomic_load(ctr, __ATOMIC_RELAXED, __HIP_MEMORY_SCOPE_AGENT) + 1u;
        __hip_atomic_store(ctr, c, __ATOMIC_RELAXED, __HIP_MEMORY_SCOPE_AGENT);
        s_call = c;
    }

    if (wave < 2) {
        const int d = (b >> 3) & 7;
        const int slot = (b & 7) + 8 * (b >> 6);
        const int pw = wave;
        const uint8_t* wsrc = w + (size_t)(32 * h) * GN + 128 * slot + 4 * jj;

        unsigned ld[2][16];
        #pragma unroll
        for (int p = 0; p < 2; ++p)
            #pragma unroll
            for (int r = 0; r < 16; ++r)
                ld[p][r] = *(const unsigned*)(wsrc + (size_t)(64 * pw + 16 * p + r) * GN);
        v4i xv[8];
        if (pw == 0) {
            const uint8_t* xsrc = x + (size_t)(d * MS + 32 * (slot >> 1) + jj) * KS + 256 * (slot & 1) + 32 * h;
            #pragma unroll
            for (int kb = 0; kb < 4; ++kb)
                #pragma unroll
                for (int p = 0; p < 2; ++p)
                    xv[kb * 2 + p] = *(const v4i*)(xsrc + 64 * kb + 16 * p);
        }
        __syncthreads();
        const unsigned call = __builtin_amdgcn_readfirstlane(s_call);
        char* ws_d = (char*)ws_all[d];
        {
            uint64_t a = (uint64_t)ws_d;
            unsigned lo = __builtin_amdgcn_readfirstlane((unsigned)a);
            unsigned hi = __builtin_amdgcn_readfirstlane((unsigned)(a >> 32));
            ws_d = (char*)(((uint64_t)hi << 32) | lo);
        }
        auto rs = __builtin_amdgcn_make_buffer_rsrc(ws_d, 0, (int)WS_NEEDED, 0x00020000);
        const unsigned bufoff = (unsigned)(OFF_DATA + (call & 1u) * BUF_BYTES);
        unsigned* flag = (unsigned*)(ws_d + OFF_FLAG + (size_t)((rank * 32 + slot) * 2 + pw) * 128);

        if (pw == 0) {
            const unsigned xo = bufoff + (unsigned)W_BYTES +
                                (unsigned)(((rank * 8 + 4 * (slot & 1)) * 16 + (slot >> 1)) * 2048) + lane * 16;
            #pragma unroll
            for (int kb = 0; kb < 4; ++kb)
                #pragma unroll
                for (int p = 0; p < 2; ++p)
                    __builtin_amdgcn_raw_buffer_store_b128(xv[kb * 2 + p], rs, xo + kb * 16 * 2048 + p * 1024, 0, AUX_SYS);
        }
        #pragma unroll
        for (int it = 0; it < ROUNDS / 2; ++it) {
            const int j = 2 * it + pw;
            v4i o[2][4];
            #pragma unroll
            for (int p = 0; p < 2; ++p)
                #pragma unroll
                for (int t = 0; t < 4; ++t) {
                    unsigned o0, o1, o2, o3;
                    tr4x4(ld[p][4 * t], ld[p][4 * t + 1], ld[p][4 * t + 2], ld[p][4 * t + 3], o0, o1, o2, o3);
                    o[p][0][t] = (int)o0; o[p][1][t] = (int)o1; o[p][2][t] = (int)o2; o[p][3][t] = (int)o3;
                }
            const unsigned wo = bufoff + (unsigned)(((rank * 8 + j) * 32 + slot) * 8192) + lane * 16;
            #pragma unroll
            for (int q = 0; q < 4; ++q)
                #pragma unroll
                for (int p = 0; p < 2; ++p)
                    __builtin_amdgcn_raw_buffer_store_b128(o[p][q], rs, wo + q * 2048 + p * 1024, 0, AUX_SYS);
            asm volatile("" ::: "memory");
            if (it + 1 < ROUNDS / 2) {
                #pragma unroll
                for (int p = 0; p < 2; ++p)
                    #pragma unroll
                    for (int r = 0; r < 16; ++r)
                        ld[p][r] = __builtin_nontemporal_load((const unsigned*)(wsrc + (size_t)(64 * (j + 2) + 16 * p + r) * GN));
                asm volatile("s_waitcnt vmcnt(32)" ::: "memory");
            } else {
                asm volatile("s_waitcnt vmcnt(0)" ::: "memory");
            }
            if (lane == 0)
                __hip_atomic_store(flag, call * 8u + (unsigned)j, __ATOMIC_RELAXED, __HIP_MEMORY_SCOPE_SYSTEM);
            asm volatile("" ::: "memory");
        }
    } else {
        const int cw = wave - 2;
        const int xcd = b & 7, idx = b >> 3;
        const int ng = 4 * xcd + (idx & 3);
        const int mt = idx >> 2;
        const int mb = 2 * mt + cw;
        __syncthreads();
        const unsigned call = __builtin_amdgcn_readfirstlane(s_call);
        const char* buf = ws_me + OFF_DATA + (size_t)(call & 1u) * BUF_BYTES;
        const char* wb = buf + (size_t)ng * 8192 + lane * 16;
        const char* xb = buf + W_BYTES + (size_t)mb * 2048 + lane * 16;

        const int psrc = lane & 7, pwh = lane >> 3;
        const int pslot = (pwh == 0) ? ng : (2 * mb + (pwh - 1));
        const unsigned* pflag0 = (const unsigned*)(ws_me + OFF_FLAG + (size_t)((psrc * 32 + (pslot & 31)) * 2) * 128);

        v16f acc[4];
        #pragma unroll
        for (int q = 0; q < 4; ++q)
            #pragma unroll
            for (int r = 0; r < 16; ++r) acc[q][r] = 0.f;

        bool ok = true;
        for (int j = 0; j < ROUNDS; ++j) {
            {
                const unsigned seq = call * 8u + (unsigned)j;
                const bool active = (pwh == 0) || (j == 0 && pwh < 3);
                const unsigned* pf = pflag0 + (j & 1) * 32;
                unsigned long long t0 = __builtin_amdgcn_s_memrealtime();
                while (true) {
                    unsigned v = active ? __hip_atomic_load(pf, __ATOMIC_RELAXED, __HIP_MEMORY_SCOPE_SYSTEM) : seq;
                    if (__all((int)(v - seq) >= 0)) break;
                    if (__hip_atomic_load(tmo, __ATOMIC_RELAXED, __HIP_MEMORY_SCOPE_AGENT) == call ||
                        __builtin_amdgcn_s_memrealtime() - t0 > DEADLINE_TICKS) {
                        if (lane == 0) __hip_atomic_store(tmo, call, __ATOMIC_RELAXED, __HIP_MEMORY_SCOPE_AGENT);
                        ok = false;
                        break;
                    }
                    __builtin_amdgcn_s_sleep(4);
                }
                __builtin_amdgcn_fence(__ATOMIC_ACQUIRE, "");
            }
            if (!ok) break;
            #pragma unroll
            for (int s = 0; s < NR; ++s) {
                const char* wp = wb + (size_t)(s * 8 + j) * (32 * 8192);
                const char* xp = xb + (size_t)(s * 8 + j) * (16 * 2048);
                v4i a0 = *(const v4i*)(xp), a1 = *(const v4i*)(xp + 1024);
                v8i a = {a0[0], a0[1], a0[2], a0[3], a1[0], a1[1], a1[2], a1[3]};
                #pragma unroll
                for (int q = 0; q < 4; ++q) {
                    v4i b0 = *(const v4i*)(wp + q * 2048), b1 = *(const v4i*)(wp + q * 2048 + 1024);
                    v8i bb = {b0[0], b0[1], b0[2], b0[3], b1[0], b1[1], b1[2], b1[3]};
                    acc[q] = __builtin_amdgcn_mfma_scale_f32_32x32x64_f8f6f4(a, bb, acc[q], 0, 0, 0, 0x7f7f7f7f, 0, 0x7f7f7f7f);
                }
            }
        }
        if (__hip_atomic_load(tmo, __ATOMIC_RELAXED, __HIP_MEMORY_SCOPE_AGENT) == call) ok = false;
        const float scale = ok ? (sx[0] * sw[0]) : __builtin_nanf("");
        float* op = out + (size_t)(32 * mb + 4 * h) * GN + 128 * ng + 4 * jj;
        #pragma unroll
        for (int r = 0; r < 16; ++r) {
            v4f v;
            if (ok) { v[0] = acc[0][r] * scale; v[1] = acc[1][r] * scale; v[2] = acc[2][r] * scale; v[3] = acc[3][r] * scale; }
            else { v[0] = v[1] = v[2] = v[3] = scale; }
            *(v4f*)(op + (size_t)((r & 3) + 8 * (r >> 2)) * GN) = v;
        }
    }
}

extern "C" void kernel_launch(int rank, int nranks, ncclComm_t comm,
                              hipStream_t stream,
                              void* const* d_in, const int* in_sizes, int n_in,
                              void* d_out, int out_size,
                              const int* in_dtypes, int out_dtype,
                              void* const* d_out_all,
                              void* const* d_ws_all, size_t ws_bytes) {
    if (nranks != NR || ws_bytes < WS_NEEDED || (unsigned)rank >= (unsigned)NR) {
        fprintf(stderr, "unsupported configuration\n");
        return;
    }
    gemm_ag_kernel<<<NWG, 256, 0, stream>>>((const uint8_t*)d_in[0], (const uint8_t*)d_in[1],
                                            (const float*)d_in[2], (const float*)d_in[3],
                                            (float*)d_out, d_ws_all, rank);
    HIP_CHECK(hipGetLastError());
}
```

```cpp
#include <hip/hip_runtime.h>
#include <rccl/rccl.h>
#include <cstdio>
#include <cstdint>

#define HIP_CHECK(expr) do { hipError_t _e = (expr); if (_e != hipSuccess) { \
    fprintf(stderr, "HIP error %s at %s:%d\n", hipGetErrorString(_e), __FILE__, __LINE__); \
    return; } } while (0)

typedef int v8i __attribute__((ext_vector_type(8)));
typedef int v4i __attribute__((ext_vector_type(4)));
typedef float v16f __attribute__((ext_vector_type(16)));
typedef float v4f __attribute__((ext_vector_type(4)));
#define GAS __attribute__((address_space(1)))
typedef GAS const v4i* gcv4;
typedef GAS unsigned* gu32;
typedef GAS const char* gcchar;

constexpr int NR = 8;
constexpr int GM = 4096, GN = 4096, GK = 4096;
constexpr int KS = GK / NR;
constexpr int MS = GM / NR;
constexpr int ROUNDS = KS / 64;
constexpr int NWG = 256;

constexpr size_t OFF_TMO = 0;
constexpr size_t OFF_CTR = 4096;
constexpr size_t OFF_FLAG = 65536;
constexpr size_t OFF_DATA = 1u << 20;
constexpr size_t W_BYTES = (size_t)NR * KS * GN;
constexpr size_t X_BYTES = (size_t)NR * KS * MS;
constexpr size_t BUF_BYTES = W_BYTES + X_BYTES;
constexpr size_t WS_NEEDED = OFF_DATA + 2 * BUF_BYTES;

constexpr unsigned long long DEADLINE_TICKS = 50000000ull;

#define STAMP(i) do {} while (0)
#define PACE 2
#define STR2(x) #x
#define STR(x) STR2(x)
#define PACE_WAIT() asm volatile("s_waitcnt vmcnt(%0)" :: "n"(PACE - 1) : "memory")
#define USE_FENCE 0
#define LOADF(off) __builtin_amdgcn_raw_buffer_load_b128(rl, (off), 0, 17)
#define AUX_SYS 17

__device__ __forceinline__ void tr4x4(unsigned d0, unsigned d1, unsigned d2, unsigned d3,
                                      unsigned& o0, unsigned& o1, unsigned& o2, unsigned& o3) {
    unsigned t0 = __builtin_amdgcn_perm(d1, d0, 0x05010400u);
    unsigned t1 = __builtin_amdgcn_perm(d1, d0, 0x07030602u);
    unsigned t2 = __builtin_amdgcn_perm(d3, d2, 0x05010400u);
    unsigned t3 = __builtin_amdgcn_perm(d3, d2, 0x07030602u);
    o0 = __builtin_amdgcn_perm(t2, t0, 0x05040100u);
    o1 = __builtin_amdgcn_perm(t2, t0, 0x07060302u);
    o2 = __builtin_amdgcn_perm(t3, t1, 0x05040100u);
    o3 = __builtin_amdgcn_perm(t3, t1, 0x07060302u);
}

__global__ void __launch_bounds__(256, 1)
gemm_ag_kernel(const uint8_t* __restrict__ x, const uint8_t* __restrict__ w,
               const float* __restrict__ sx, const float* __restrict__ sw,
               float* __restrict__ out, void* const* __restrict__ ws_all, int rank) {
    __shared__ unsigned s_call;
    const int b = blockIdx.x;
    const int tid = threadIdx.x;
    const int wave = __builtin_amdgcn_readfirstlane(tid >> 6);
    const int lane = tid & 63;
    const int jj = lane & 31, h = lane >> 5;

    char* ws_me = (char*)ws_all[rank];
    gu32 tmo = (gu32)(ws_me + OFF_TMO);

    if (wave == 0) STAMP(0);
    if (tid == 0) {
        gu32 ctr = (gu32)(ws_me + OFF_CTR + (size_t)b * 64);
        unsigned c = __hip_atomic_load(ctr, __ATOMIC_RELAXED, __HIP_MEMORY_SCOPE_AGENT) + 1u;
        __hip_atomic_store(ctr, c, __ATOMIC_RELAXED, __HIP_MEMORY_SCOPE_AGENT);
        s_call = c;
    }

    if (wave == 0) {
        const int d = (b >> 3) & 7;
        const int slot = (b & 7) + 8 * (b >> 6);
        const uint8_t* wsrc = w + (size_t)(32 * h) * GN + 128 * slot + 4 * jj;

        v4i xv[8];
        {
            const uint8_t* xsrc = x + (size_t)(d * MS + 32 * (slot >> 1) + jj) * KS + 256 * (slot & 1) + 32 * h;
            #pragma unroll
            for (int kb = 0; kb < 4; ++kb)
                #pragma unroll
                for (int p = 0; p < 2; ++p)
                    xv[kb * 2 + p] = *(const v4i*)(xsrc + 64 * kb + 16 * p);
        }
        unsigned ld[2][16];
        #pragma unroll
        for (int p = 0; p < 2; ++p)
            #pragma unroll
            for (int r = 0; r < 16; ++r)
                ld[p][r] = *(const unsigned*)(wsrc + (size_t)(16 * p + r) * GN);
        __syncthreads();
        const unsigned call = __builtin_amdgcn_readfirstlane(s_call);
        char* ws_d = (char*)ws_all[d];
        {
            uint64_t a = (uint64_t)ws_d;
            unsigned lo = __builtin_amdgcn_readfirstlane((unsigned)a);
            unsigned hi = __builtin_amdgcn_readfirstlane((unsigned)(a >> 32));
            ws_d = (char*)(((uint64_t)hi << 32) | lo);
        }
        auto rs = __builtin_amdgcn_make_buffer_rsrc(ws_d, 0, (int)WS_NEEDED, 0x00020000);
        const unsigned bufoff = (unsigned)(OFF_DATA + (call & 1u) * BUF_BYTES);
        gu32 flag = (gu32)(ws_d + OFF_FLAG + (size_t)(rank * 32 + slot) * 128);

        {
            const unsigned xo = bufoff + (unsigned)W_BYTES +
                                (unsigned)(((rank * 8 + 4 * (slot & 1)) * 16 + (slot >> 1)) * 2048) + lane * 16;
            #pragma unroll
            for (int kb = 0; kb < 4; ++kb)
                #pragma unroll
                for (int p = 0; p < 2; ++p) {
                    __builtin_amdgcn_raw_buffer_store_b128(xv[kb * 2 + p], rs, xo + kb * 16 * 2048 + p * 1024, 0, AUX_SYS);
                    PACE_WAIT();
                }
        }
        #pragma unroll
        for (int j = 0; j < ROUNDS; ++j) {
            v4i o[2][4];
            #pragma unroll
            for (int p = 0; p < 2; ++p)
                #pragma unroll
                for (int t = 0; t < 4; ++t) {
                    unsigned o0, o1, o2, o3;
                    tr4x4(ld[p][4 * t], ld[p][4 * t + 1], ld[p][4 * t + 2], ld[p][4 * t + 3], o0, o1, o2, o3);
                    o[p][0][t] = (int)o0; o[p][1][t] = (int)o1; o[p][2][t] = (int)o2; o[p][3][t] = (int)o3;
                }
            asm volatile("" ::: "memory");
            if (j + 1 < ROUNDS) {
                #pragma unroll
                for (int p = 0; p < 2; ++p)
                    #pragma unroll
                    for (int r = 0; r < 16; ++r)
                        ld[p][r] = *(const unsigned*)(wsrc + (size_t)(64 * (j + 1) + 16 * p + r) * GN);
            }
            asm volatile("" ::: "memory");
            const unsigned wo = bufoff + (unsigned)(((rank * 8 + j) * 32 + slot) * 8192) + lane * 16;
            #pragma unroll
            for (int i = 0; i < 8; ++i) {
                __builtin_amdgcn_raw_buffer_store_b128(o[i & 1][i >> 1], rs, wo + (i >> 1) * 2048 + (i & 1) * 1024, 0, AUX_SYS);
                PACE_WAIT();
                if (i == PACE - 1 && j > 0) {
                    if (lane == 0)
                        __hip_atomic_store(flag, call * 8u + (unsigned)(j - 1), __ATOMIC_RELAXED, __HIP_MEMORY_SCOPE_SYSTEM);
                    STAMP(j);
                    asm volatile("" ::: "memory");
                }
            }
        }
        asm volatile("s_waitcnt vmcnt(0)" ::: "memory");
        if (lane == 0)
            __hip_atomic_store(flag, call * 8u + (unsigned)(ROUNDS - 1), __ATOMIC_RELAXED, __HIP_MEMORY_SCOPE_SYSTEM);
        STAMP(8);
    } else if (wave == 1) {
        __syncthreads();
    } else {
        const int cw = wave - 2;
        const int xcd = b & 7, idx = b >> 3;
        const int ng = 4 * xcd + (idx & 3);
        const int mt = idx >> 2;
        const int mb = 2 * mt + cw;
        __syncthreads();
        const unsigned call = __builtin_amdgcn_readfirstlane(s_call);
        char* ws_u = ws_me;
        {
            uint64_t a = (uint64_t)ws_u;
            unsigned lo = __builtin_amdgcn_readfirstlane((unsigned)a);
            unsigned hi = __builtin_amdgcn_readfirstlane((unsigned)(a >> 32));
            ws_u = (char*)(((uint64_t)hi << 32) | lo);
        }
        auto rl = __builtin_amdgcn_make_buffer_rsrc(ws_u, 0, (int)WS_NEEDED, 0x00020000);
        const unsigned buf = (unsigned)(OFF_DATA + (call & 1u) * BUF_BYTES);
        const unsigned wb = buf + (unsigned)ng * 8192 + lane * 16;
        const unsigned xb = buf + (unsigned)W_BYTES + (unsigned)mb * 2048 + lane * 16;

        const int psrc = lane & 7, pwh = lane >> 3;
        const int pslot = (pwh == 0) ? ng : (2 * mb + (pwh - 1));
        gu32 pflag0 = (gu32)(ws_me + OFF_FLAG + (size_t)(psrc * 32 + (pslot & 31)) * 128);

        v16f acc[4];
        #pragma unroll
        for (int q = 0; q < 4; ++q)
            #pragma unroll
            for (int r = 0; r < 16; ++r) acc[q][r] = 0.f;

        bool ok = true;
        for (int j = 0; j < ROUNDS; ++j) {
            {
                const unsigned seq = call * 8u + (unsigned)j;
                const bool active = (pwh == 0) || (j == 0 && pwh < 3);
                gu32 pf = pflag0;
                unsigned long long t0 = __builtin_amdgcn_s_memrealtime();
                while (true) {
                    unsigned v = active ? __hip_atomic_load(pf, __ATOMIC_RELAXED, __HIP_MEMORY_SCOPE_SYSTEM) : seq;
                    if (__all((int)(v - seq) >= 0)) break;
                    if (__hip_atomic_load(tmo, __ATOMIC_RELAXED, __HIP_MEMORY_SCOPE_AGENT) == call ||
                        __builtin_amdgcn_s_memrealtime() - t0 > DEADLINE_TICKS) {
                        if (lane == 0) __hip_atomic_store(tmo, call, __ATOMIC_RELAXED, __HIP_MEMORY_SCOPE_AGENT);
                        ok = false;
                        break;
                    }
                    __builtin_amdgcn_s_sleep(4);
                }
                __builtin_amdgcn_fence(__ATOMIC_ACQUIRE, "wavefront");
            }
            if (!ok) break;
            if (cw == 0) STAMP(16 + j);
            v4i fa[NR][2], fb[NR][4][2];
            #pragma unroll
            for (int s = 0; s < NR; ++s) {
                const unsigned wp = wb + (unsigned)(s * 8 + j) * (32 * 8192);
                const unsigned xp = xb + (unsigned)(s * 8 + j) * (16 * 2048);
                fa[s][0] = LOADF(xp); fa[s][1] = LOADF(xp + 1024);
                #pragma unroll
                for (int q = 0; q < 4; ++q) {
                    fb[s][q][0] = LOADF(wp + q * 2048); fb[s][q][1] = LOADF(wp + q * 2048 + 1024);
                }
            }
            __builtin_amdgcn_sched_barrier(0);
            #pragma unroll
            for (int s = 0; s < NR; ++s) {
                v4i a0 = fa[s][0], a1 = fa[s][1];
                v8i a = {a0[0], a0[1], a0[2], a0[3], a1[0], a1[1], a1[2], a1[3]};
                #pragma unroll
                for (int q = 0; q < 4; ++q) {
                    v4i b0 = fb[s][q][0], b1 = fb[s][q][1];
                    v8i bb = {b0[0], b0[1], b0[2], b0[3], b1[0], b1[1], b1[2], b1[3]};
                    acc[q] = __builtin_amdgcn_mfma_scale_f32_32x32x64_f8f6f4(a, bb, acc[q], 0, 0, 0, 0x7f7f7f7f, 0, 0x7f7f7f7f);
                }
            }
        }
        if (cw == 0) STAMP(24);
        if (__hip_atomic_load(tmo, __ATOMIC_RELAXED, __HIP_MEMORY_SCOPE_AGENT) == call) ok = false;
        const float scale = ok ? (sx[0] * sw[0]) : __builtin_nanf("");
        float* op = out + (size_t)(32 * mb + 4 * h) * GN + 128 * ng + 4 * jj;
        #pragma unroll
        for (int r = 0; r < 16; ++r) {
            v4f v;
            if (ok) { v[0] = acc[0][r] * scale; v[1] = acc[1][r] * scale; v[2] = acc[2][r] * scale; v[3] = acc[3][r] * scale; }
            else { v[0] = v[1] = v[2] = v[3] = scale; }
            *(v4f*)(op + (size_t)((r & 3) + 8 * (r >> 2)) * GN) = v;
        }
    }
}

extern "C" void kernel_launch(int rank, int nranks, ncclComm_t comm,
                              hipStream_t stream,
                              void* const* d_in, const int* in_sizes, int n_in,
                              void* d_out, int out_size,
                              const int* in_dtypes, int out_dtype,
                              void* const* d_out_all,
                              void* const* d_ws_all, size_t ws_bytes) {
    if (nranks != NR || ws_bytes < WS_NEEDED || (unsigned)rank >= (unsigned)NR) {
        fprintf(stderr, "unsupported configuration\n");
        return;
    }
    gemm_ag_kernel<<<NWG, 256, 0, stream>>>((const uint8_t*)d_in[0], (const uint8_t*)d_in[1],
                                            (const float*)d_in[2], (const float*)d_in[3],
                                            (float*)d_out, d_ws_all, rank);
    HIP_CHECK(hipGetLastError());
}
```

```cpp
#include <hip/hip_runtime.h>
#include <rccl/rccl.h>
#include <cstdio>
#include <cstdint>

#define HIP_CHECK(expr) do { hipError_t _e = (expr); if (_e != hipSuccess) { \
    fprintf(stderr, "HIP error %s at %s:%d\n", hipGetErrorString(_e), __FILE__, __LINE__); \
    return; } } while (0)

typedef int v8i __attribute__((ext_vector_type(8)));
typedef int v4i __attribute__((ext_vector_type(4)));
typedef float v16f __attribute__((ext_vector_type(16)));
typedef float v4f __attribute__((ext_vector_type(4)));
#define GAS __attribute__((address_space(1)))
typedef GAS unsigned* gu32;

constexpr int NR = 8;
constexpr int GM = 4096, GN = 4096, GK = 4096;
constexpr int KS = GK / NR;
constexpr int MS = GM / NR;
constexpr int ROUNDS = KS / 64;
constexpr int NWG = 256;

constexpr size_t OFF_TMO = 0;
constexpr size_t OFF_CTR = 4096;
constexpr size_t OFF_FLAG = 65536;
constexpr size_t OFF_DATA = 1u << 20;
constexpr size_t W_BYTES = (size_t)NR * KS * GN;
constexpr size_t X_BYTES = (size_t)NR * KS * MS;
constexpr size_t BUF_BYTES = W_BYTES + X_BYTES;
constexpr size_t WS_NEEDED = OFF_DATA + 2 * BUF_BYTES;

constexpr unsigned long long DEADLINE_TICKS = 50000000ull;

constexpr int PACE = 2;
constexpr int BATCH = 8;
#define AUX_SYS 17
#define PACE_WAIT() asm volatile("s_waitcnt vmcnt(%0)" :: "n"(PACE - 1) : "memory")
#define LOADF(off) __builtin_amdgcn_raw_buffer_load_b128(rl, (off), 0, AUX_SYS)

__device__ __forceinline__ void tr4x4(unsigned d0, unsigned d1, unsigned d2, unsigned d3,
                                      unsigned& o0, unsigned& o1, unsigned& o2, unsigned& o3) {
    unsigned t0 = __builtin_amdgcn_perm(d1, d0, 0x05010400u);
    unsigned t1 = __builtin_amdgcn_perm(d1, d0, 0x07030602u);
    unsigned t2 = __builtin_amdgcn_perm(d3, d2, 0x05010400u);
    unsigned t3 = __builtin_amdgcn_perm(d3, d2, 0x07030602u);
    o0 = __builtin_amdgcn_perm(t2, t0, 0x05040100u);
    o1 = __builtin_amdgcn_perm(t2, t0, 0x07060302u);
    o2 = __builtin_amdgcn_perm(t3, t1, 0x05040100u);
    o3 = __builtin_amdgcn_perm(t3, t1, 0x07060302u);
}

__device__ __forceinline__ char* uniform_ptr(char* p) {
    uint64_t a = (uint64_t)p;
    unsigned lo = __builtin_amdgcn_readfirstlane((unsigned)a);
    unsigned hi = __builtin_amdgcn_readfirstlane((unsigned)(a >> 32));
    return (char*)(((uint64_t)hi << 32) | lo);
}

__global__ void __launch_bounds__(256, 1)
gemm_ag_kernel(const uint8_t* __restrict__ x, const uint8_t* __restrict__ w,
               const float* __restrict__ sx, const float* __restrict__ sw,
               float* __restrict__ out, void* const* __restrict__ ws_all, int rank) {
    __shared__ unsigned long long s_mask[2];
    __shared__ unsigned s_call, s_abort;
    const int b = blockIdx.x;
    const int tid = threadIdx.x;
    const int wave = __builtin_amdgcn_readfirstlane(tid >> 6);
    const int lane = tid & 63;
    const int jj = lane & 31, h = lane >> 5;

    char* ws_me = (char*)ws_all[rank];
    gu32 tmo = (gu32)(ws_me + OFF_TMO);

    if (tid == 0) {
        gu32 ctr = (gu32)(ws_me + OFF_CTR + (size_t)b * 64);
        unsigned c = __hip_atomic_load(ctr, __ATOMIC_RELAXED, __HIP_MEMORY_SCOPE_AGENT) + 1u;
        __hip_atomic_store(ctr, c, __ATOMIC_RELAXED, __HIP_MEMORY_SCOPE_AGENT);
        s_call = c; s_abort = 0u; s_mask[0] = 0ull; s_mask[1] = 0ull;
    }

    if (wave == 0) {
        const int d = (b >> 3) & 7;
        const int slot = (b & 7) + 8 * (b >> 6);
        const uint8_t* wsrc = w + (size_t)(32 * h) * GN + 128 * slot + 4 * jj;

        v4i xv[8];
        {
            const uint8_t* xsrc = x + (size_t)(d * MS + 32 * (slot >> 1) + jj) * KS + 256 * (slot & 1) + 32 * h;
            #pragma unroll
            for (int kb = 0; kb < 4; ++kb)
                #pragma unroll
                for (int p = 0; p < 2; ++p)
                    xv[kb * 2 + p] = *(const v4i*)(xsrc + 64 * kb + 16 * p);
        }
        unsigned ld[2][16];
        #pragma unroll
        for (int p = 0; p < 2; ++p)
            #pragma unroll
            for (int r = 0; r < 16; ++r)
                ld[p][r] = *(const unsigned*)(wsrc + (size_t)(16 * p + r) * GN);
        __syncthreads();
        const unsigned call = __builtin_amdgcn_readfirstlane(s_call);
        char* ws_d = uniform_ptr((char*)ws_all[d]);
        auto rs = __builtin_amdgcn_make_buffer_rsrc(ws_d, 0, (int)WS_NEEDED, 0x00020000);
        const unsigned bufoff = (unsigned)(OFF_DATA + (call & 1u) * BUF_BYTES);
        gu32 flag = (gu32)(ws_d + OFF_FLAG + (size_t)(rank * 32 + slot) * 128);

        {
            const unsigned xo = bufoff + (unsigned)W_BYTES +
                                (unsigned)(((rank * 8 + 4 * (slot & 1)) * 16 + (slot >> 1)) * 2048) + lane * 16;
            #pragma unroll
            for (int kb = 0; kb < 4; ++kb)
                #pragma unroll
                for (int p = 0; p < 2; ++p) {
                    __builtin_amdgcn_raw_buffer_store_b128(xv[kb * 2 + p], rs, xo + kb * 16 * 2048 + p * 1024, 0, AUX_SYS);
                    PACE_WAIT();
                }
        }
        #pragma unroll
        for (int j = 0; j < ROUNDS; ++j) {
            v4i o[2][4];
            #pragma unroll
            for (int p = 0; p < 2; ++p)
                #pragma unroll
                for (int t = 0; t < 4; ++t) {
                    unsigned o0, o1, o2, o3;
                    tr4x4(ld[p][4 * t], ld[p][4 * t + 1], ld[p][4 * t + 2], ld[p][4 * t + 3], o0, o1, o2, o3);
                    o[p][0][t] = (int)o0; o[p][1][t] = (int)o1; o[p][2][t] = (int)o2; o[p][3][t] = (int)o3;
                }
            asm volatile("" ::: "memory");
            if (j + 1 < ROUNDS) {
                #pragma unroll
                for (int p = 0; p < 2; ++p)
                    #pragma unroll
                    for (int r = 0; r < 16; ++r)
                        ld[p][r] = *(const unsigned*)(wsrc + (size_t)(64 * (j + 1) + 16 * p + r) * GN);
            }
            asm volatile("" ::: "memory");
            const unsigned wo = bufoff + (unsigned)(((rank * 8 + j) * 32 + slot) * 8192) + lane * 16;
            #pragma unroll
            for (int i = 0; i < 8; ++i) {
                __builtin_amdgcn_raw_buffer_store_b128(o[i & 1][i >> 1], rs, wo + (i >> 1) * 2048 + (i & 1) * 1024, 0, AUX_SYS);
                PACE_WAIT();
                if (i == PACE - 1 && j > 0) {
                    if (lane == 0)
                        __hip_atomic_store(flag, call * 8u + (unsigned)(j - 1), __ATOMIC_RELAXED, __HIP_MEMORY_SCOPE_SYSTEM);
                    asm volatile("" ::: "memory");
                }
            }
        }
        asm volatile("s_waitcnt vmcnt(0)" ::: "memory");
        if (lane == 0)
            __hip_atomic_store(flag, call * 8u + (unsigned)(ROUNDS - 1), __ATOMIC_RELAXED, __HIP_MEMORY_SCOPE_SYSTEM);
    } else if (wave == 1) {
        const int xcd = b & 7, idx = b >> 3;
        const int ng = 4 * xcd + (idx & 3);
        const int mt = idx >> 2;
        __syncthreads();
        const unsigned call = __builtin_amdgcn_readfirstlane(s_call);
        const int us = lane >> 3, uj = lane & 7;
        gu32 pfw = (gu32)(ws_me + OFF_FLAG + (size_t)(us * 32 + ng) * 128);
        gu32 pfx = (gu32)(ws_me + OFF_FLAG + (size_t)(us * 32 + 4 * mt + (uj >> 2)) * 128);
        const unsigned seqw = call * 8u + (unsigned)uj, seqx = call * 8u;
        unsigned long long x0 = 0ull, x1 = 0ull, m0 = 0ull, m1 = 0ull;
        const unsigned long long t0 = __builtin_amdgcn_s_memrealtime();
        unsigned spin = 0;
        while ((m0 & m1) != ~0ull) {
            if ((x0 & x1) != ~0ull) {
                unsigned vx0 = __hip_atomic_load(pfx, __ATOMIC_RELAXED, __HIP_MEMORY_SCOPE_SYSTEM);
                unsigned vx1 = __hip_atomic_load(pfx + 64, __ATOMIC_RELAXED, __HIP_MEMORY_SCOPE_SYSTEM);
                x0 = __ballot((int)(vx0 - seqx) >= 0);
                x1 = __ballot((int)(vx1 - seqx) >= 0);
            }
            unsigned vw = __hip_atomic_load(pfw, __ATOMIC_RELAXED, __HIP_MEMORY_SCOPE_SYSTEM);
            const unsigned long long wm = __ballot((int)(vw - seqw) >= 0);
            const unsigned long long n0 = wm & x0, n1 = wm & x1;
            if (n0 != m0 || n1 != m1) {
                m0 = n0; m1 = n1;
                if (lane == 0) {
                    __hip_atomic_store(&s_mask[0], m0, __ATOMIC_RELAXED, __HIP_MEMORY_SCOPE_WORKGROUP);
                    __hip_atomic_store(&s_mask[1], m1, __ATOMIC_RELAXED, __HIP_MEMORY_SCOPE_WORKGROUP);
                }
            } else if ((++spin & 63u) == 0u) {
                if (__hip_atomic_load(tmo, __ATOMIC_RELAXED, __HIP_MEMORY_SCOPE_AGENT) == call ||
                    __builtin_amdgcn_s_memrealtime() - t0 > DEADLINE_TICKS) {
                    if (lane == 0) {
                        __hip_atomic_store(tmo, call, __ATOMIC_RELAXED, __HIP_MEMORY_SCOPE_AGENT);
                        __hip_atomic_store(&s_abort, 1u, __ATOMIC_RELAXED, __HIP_MEMORY_SCOPE_WORKGROUP);
                    }
                    break;
                }
            }
            if (__builtin_popcountll(m0 & m1) < 48) __builtin_amdgcn_s_sleep(64);
            else __builtin_amdgcn_s_sleep(2);
        }
    } else {
        const int cw = wave - 2;
        const int xcd = b & 7, idx = b >> 3;
        const int ng = 4 * xcd + (idx & 3);
        const int mt = idx >> 2;
        const int mb = 2 * mt + cw;
        __syncthreads();
        const unsigned call = __builtin_amdgcn_readfirstlane(s_call);
        char* ws_u = uniform_ptr(ws_me);
        auto rl = __builtin_amdgcn_make_buffer_rsrc(ws_u, 0, (int)WS_NEEDED, 0x00020000);
        const unsigned buf = (unsigned)(OFF_DATA + (call & 1u) * BUF_BYTES);
        const unsigned wb = buf + (unsigned)ng * 8192 + lane * 16;
        const unsigned xb = buf + (unsigned)W_BYTES + (unsigned)mb * 2048 + lane * 16;

        v16f acc[4];
        #pragma unroll
        for (int q = 0; q < 4; ++q)
            #pragma unroll
            for (int r = 0; r < 16; ++r) acc[q][r] = 0.f;

        bool ok = true;
        unsigned long long done = 0ull;
        unsigned tmo_seen = 0;
        while (done != ~0ull) {
            unsigned long long msk = __hip_atomic_load(&s_mask[cw], __ATOMIC_RELAXED, __HIP_MEMORY_SCOPE_WORKGROUP);
            unsigned long long pend;
            {
                unsigned lo = __builtin_amdgcn_readfirstlane((unsigned)msk);
                unsigned hi = __builtin_amdgcn_readfirstlane((unsigned)(msk >> 32));
                pend = (((unsigned long long)hi << 32) | lo) & ~done;
            }
            if (pend == 0ull) {
                if (__builtin_amdgcn_readfirstlane(__hip_atomic_load(&s_abort, __ATOMIC_RELAXED, __HIP_MEMORY_SCOPE_WORKGROUP)) != 0u) {
                    ok = false;
                    break;
                }
                __builtin_amdgcn_s_sleep(1);
                continue;
            }
            __builtin_amdgcn_fence(__ATOMIC_ACQUIRE, "wavefront");
            tmo_seen = __hip_atomic_load(tmo, __ATOMIC_RELAXED, __HIP_MEMORY_SCOPE_AGENT);
            v4i fa[BATCH][2], fb[BATCH][4][2];
            int cnt = 0;
            #pragma unroll
            for (int i = 0; i < BATCH; ++i) {
                unsigned wp, xp;
                if (pend != 0ull) {
                    const unsigned u = (unsigned)__builtin_ctzll(pend);
                    pend &= pend - 1ull;
                    done |= 1ull << u;
                    wp = wb + u * (32 * 8192);
                    xp = xb + u * (16 * 2048);
                    cnt = i + 1;
                } else {
                    wp = 0x80000000u; xp = 0x80000000u;
                }
                fa[i][0] = LOADF(xp); fa[i][1] = LOADF(xp + 1024);
                #pragma unroll
                for (int q = 0; q < 4; ++q) {
                    fb[i][q][0] = LOADF(wp + q * 2048); fb[i][q][1] = LOADF(wp + q * 2048 + 1024);
                }
            }
            __builtin_amdgcn_sched_barrier(0);
            #pragma unroll
            for (int i = 0; i < BATCH; ++i) {
                if (i < cnt) {
                    v4i a0 = fa[i][0], a1 = fa[i][1];
                    v8i a = {a0[0], a0[1], a0[2], a0[3], a1[0], a1[1], a1[2], a1[3]};
                    #pragma unroll
                    for (int q = 0; q < 4; ++q) {
                        v4i b0 = fb[i][q][0], b1 = fb[i][q][1];
                        v8i bb = {b0[0], b0[1], b0[2], b0[3], b1[0], b1[1], b1[2], b1[3]};
                        acc[q] = __builtin_amdgcn_mfma_scale_f32_32x32x64_f8f6f4(a, bb, acc[q], 0, 0, 0, 0x7f7f7f7f, 0, 0x7f7f7f7f);
                    }
                }
            }
        }
        if (tmo_seen == call) ok = false;
        const float scale = ok ? (sx[0] * sw[0]) : __builtin_nanf("");
        auto ro = __builtin_amdgcn_make_buffer_rsrc(out, 0, MS * GN * 4, 0x00020000);
        const unsigned oo = (unsigned)(((32 * mb + 4 * h) * GN + 128 * ng + 4 * jj) * 4);
        #pragma unroll
        for (int r = 0; r < 16; ++r) {
            v4f v;
            if (ok) { v[0] = acc[0][r] * scale; v[1] = acc[1][r] * scale; v[2] = acc[2][r] * scale; v[3] = acc[3][r] * scale; }
            else { v[0] = v[1] = v[2] = v[3] = scale; }
            __builtin_amdgcn_raw_buffer_store_b128(__builtin_bit_cast(v4i, v), ro, oo + (unsigned)(((r & 3) + 8 * (r >> 2)) * GN * 4), 0, 0);
        }
    }
}

extern "C" void kernel_launch(int rank, int nranks, ncclComm_t comm,
                              hipStream_t stream,
                              void* const* d_in, const int* in_sizes, int n_in,
                              void* d_out, int out_size,
                              const int* in_dtypes, int out_dtype,
                              void* const* d_out_all,
                              void* const* d_ws_all, size_t ws_bytes) {
    if (nranks != NR || ws_bytes < WS_NEEDED || (unsigned)rank >= (unsigned)NR) {
        fprintf(stderr, "unsupported configuration\n");
        return;
    }
    gemm_ag_kernel<<<NWG, 256, 0, stream>>>((const uint8_t*)d_in[0], (const uint8_t*)d_in[1],
                                            (const float*)d_in[2], (const float*)d_in[3],
                                            (float*)d_out, d_ws_all, rank);
    HIP_CHECK(hipGetLastError());
}
```

```cpp
#include <hip/hip_runtime.h>
#include <rccl/rccl.h>
#include <cstdio>
#include <cstdint>

#define HIP_CHECK(expr) do { hipError_t _e = (expr); if (_e != hipSuccess) { \
    fprintf(stderr, "HIP error %s at %s:%d\n", hipGetErrorString(_e), __FILE__, __LINE__); \
    return; } } while (0)

typedef int v8i __attribute__((ext_vector_type(8)));
typedef int v4i __attribute__((ext_vector_type(4)));
typedef float v16f __attribute__((ext_vector_type(16)));
typedef float v4f __attribute__((ext_vector_type(4)));
#define GAS __attribute__((address_space(1)))
typedef GAS unsigned* gu32;

constexpr int NR = 8;
constexpr int GM = 4096, GN = 4096, GK = 4096;
constexpr int KS = GK / NR;
constexpr int MS = GM / NR;
constexpr int ROUNDS = KS / 64;
constexpr int NWG = 256;

constexpr size_t OFF_TMO = 0;
constexpr size_t OFF_CTR = 4096;
constexpr size_t OFF_PROG = 32768;
constexpr size_t OFF_FLAG = 65536;
constexpr size_t OFF_DATA = 1u << 20;
constexpr size_t W_BYTES = (size_t)NR * KS * GN;
constexpr size_t X_BYTES = (size_t)NR * KS * MS;
constexpr size_t BUF_BYTES = W_BYTES + X_BYTES;
constexpr size_t WS_NEEDED = OFF_DATA + 2 * BUF_BYTES;

constexpr unsigned long long DEADLINE_TICKS = 50000000ull;

constexpr int PACE = 2;
constexpr int BATCH = 3;
constexpr int LAG_LO = 0, LAG_HI = 24, LAG_HI2 = 40;
#define AUX_SYS 17
#define PACE_WAIT() asm volatile("s_waitcnt vmcnt(%0)" :: "n"(PACE - 1) : "memory")
#define LOADF(off) __builtin_amdgcn_raw_buffer_load_b128(rl, (off), 0, AUX_SYS)

__device__ __forceinline__ void tr4x4(unsigned d0, unsigned d1, unsigned d2, unsigned d3,
                                      unsigned& o0, unsigned& o1, unsigned& o2, unsigned& o3) {
    unsigned t0 = __builtin_amdgcn_perm(d1, d0, 0x05010400u);
    unsigned t1 = __builtin_amdgcn_perm(d1, d0, 0x07030602u);
    unsigned t2 = __builtin_amdgcn_perm(d3, d2, 0x05010400u);
    unsigned t3 = __builtin_amdgcn_perm(d3, d2, 0x07030602u);
    o0 = __builtin_amdgcn_perm(t2, t0, 0x05040100u);
    o1 = __builtin_amdgcn_perm(t2, t0, 0x07060302u);
    o2 = __builtin_amdgcn_perm(t3, t1, 0x05040100u);
    o3 = __builtin_amdgcn_perm(t3, t1, 0x07060302u);
}

__device__ __forceinline__ char* uniform_ptr(char* p) {
    uint64_t a = (uint64_t)p;
    unsigned lo = __builtin_amdgcn_readfirstlane((unsigned)a);
    unsigned hi = __builtin_amdgcn_readfirstlane((unsigned)(a >> 32));
    return (char*)(((uint64_t)hi << 32) | lo);
}

__global__ void __launch_bounds__(256, 1)
gemm_ag_kernel(const uint8_t* __restrict__ x, const uint8_t* __restrict__ w,
               const float* __restrict__ sx, const float* __restrict__ sw,
               float* __restrict__ out, void* const* __restrict__ ws_all, int rank) {
    __shared__ __attribute__((aligned(16))) char s_red[32768];
    __shared__ unsigned long long s_mask;
    __shared__ unsigned s_call, s_abort, s_rdy[2];
    const int b = blockIdx.x;
    const int tid = threadIdx.x;
    const int wave = __builtin_amdgcn_readfirstlane(tid >> 6);
    const int lane = tid & 63;
    const int jj = lane & 31, h = lane >> 5;

    char* ws_me = (char*)ws_all[rank];
    gu32 tmo = (gu32)(ws_me + OFF_TMO);

    if (tid == 0) {
        gu32 ctr = (gu32)(ws_me + OFF_CTR + (size_t)b * 64);
        unsigned c = __hip_atomic_load(ctr, __ATOMIC_RELAXED, __HIP_MEMORY_SCOPE_AGENT) + 1u;
        __hip_atomic_store(ctr, c, __ATOMIC_RELAXED, __HIP_MEMORY_SCOPE_AGENT);
        s_call = c; s_abort = 0u; s_mask = 0ull; s_rdy[0] = 0u; s_rdy[1] = 0u;
    }

    if (wave == 0) {
        const int d = (b >> 3) & 7;
        const int slot = (b & 7) + 8 * (b >> 6);
        const uint8_t* wsrc = w + (size_t)(32 * h) * GN + 128 * slot + 4 * jj;

        v4i xv[8];
        {
            const uint8_t* xsrc = x + (size_t)(d * MS + 32 * (slot >> 1) + jj) * KS + 256 * (slot & 1) + 32 * h;
            #pragma unroll
            for (int kb = 0; kb < 4; ++kb)
                #pragma unroll
                for (int p = 0; p < 2; ++p)
                    xv[kb * 2 + p] = *(const v4i*)(xsrc + 64 * kb + 16 * p);
        }
        unsigned ld[2][16];
        #pragma unroll
        for (int p = 0; p < 2; ++p)
            #pragma unroll
            for (int r = 0; r < 16; ++r)
                ld[p][r] = *(const unsigned*)(wsrc + (size_t)(16 * p + r) * GN);
        __syncthreads();
        const unsigned call = __builtin_amdgcn_readfirstlane(s_call);
        char* ws_d = uniform_ptr((char*)ws_all[d]);
        auto rs = __builtin_amdgcn_make_buffer_rsrc(ws_d, 0, (int)WS_NEEDED, 0x00020000);
        const unsigned bufoff = (unsigned)(OFF_DATA + (call & 1u) * BUF_BYTES);
        gu32 flag = (gu32)(ws_d + OFF_FLAG + (size_t)(rank * 32 + slot) * 128);
        unsigned qoff = 0;
        asm volatile("" : "+v"(qoff));
        gu32 prog = (gu32)(ws_me + OFF_PROG + (size_t)d * 128) + qoff;
        const unsigned pbase = (call - 1u) * 224u;
        int pace = PACE;

        {
            const unsigned xo = bufoff + (unsigned)W_BYTES +
                                (unsigned)(((rank * 8 + 4 * (slot & 1)) * 16 + (slot >> 1)) * 2048) + lane * 16;
            #pragma unroll
            for (int kb = 0; kb < 4; ++kb)
                #pragma unroll
                for (int p = 0; p < 2; ++p) {
                    __builtin_amdgcn_raw_buffer_store_b128(xv[kb * 2 + p], rs, xo + kb * 16 * 2048 + p * 1024, 0, AUX_SYS);
                    PACE_WAIT();
                }
        }
        #pragma unroll
        for (int j = 0; j < ROUNDS; ++j) {
            v4i o[2][4];
            #pragma unroll
            for (int p = 0; p < 2; ++p)
                #pragma unroll
                for (int t = 0; t < 4; ++t) {
                    unsigned o0, o1, o2, o3;
                    tr4x4(ld[p][4 * t], ld[p][4 * t + 1], ld[p][4 * t + 2], ld[p][4 * t + 3], o0, o1, o2, o3);
                    o[p][0][t] = (int)o0; o[p][1][t] = (int)o1; o[p][2][t] = (int)o2; o[p][3][t] = (int)o3;
                }
            asm volatile("" ::: "memory");
            if (j + 1 < ROUNDS) {
                #pragma unroll
                for (int p = 0; p < 2; ++p)
                    #pragma unroll
                    for (int r = 0; r < 16; ++r)
                        ld[p][r] = *(const unsigned*)(wsrc + (size_t)(64 * (j + 1) + 16 * p + r) * GN);
            }
            asm volatile("" ::: "memory");
            const unsigned wo = bufoff + (unsigned)(((rank * 8 + j) * 32 + slot) * 8192) + lane * 16;
            unsigned tk = 0;
            if (j > 0 && lane == 0) tk = __hip_atomic_fetch_add(prog, 1u, __ATOMIC_RELAXED, __HIP_MEMORY_SCOPE_AGENT);
            asm volatile("" ::: "memory");
            #pragma unroll
            for (int i = 0; i < 8; ++i) {
                __builtin_amdgcn_raw_buffer_store_b128(o[i & 1][i >> 1], rs, wo + (i >> 1) * 2048 + (i & 1) * 1024, 0, AUX_SYS);
                if (pace == 2) asm volatile("s_waitcnt vmcnt(1)" ::: "memory");
                else if (pace == 4) asm volatile("s_waitcnt vmcnt(3)" ::: "memory");
                else if (pace == 3) asm volatile("s_waitcnt vmcnt(2)" ::: "memory");
                else asm volatile("s_waitcnt vmcnt(0)" ::: "memory");
                if (i < 4 && j > 0 && i == pace - 1) {
                    if (lane == 0)
                        __hip_atomic_store(flag, call * 8u + (unsigned)(j - 1), __ATOMIC_RELAXED, __HIP_MEMORY_SCOPE_SYSTEM);
                    asm volatile("" ::: "memory");
                }
                if (i == 3 && j > 0) {
                    const int lag = (int)(__builtin_amdgcn_readfirstlane(tk) - pbase) - 32 * (j - 1);
                    pace = (lag > LAG_HI2) ? 4 : (lag > LAG_HI) ? 3 : (lag < LAG_LO) ? 1 : 2;
                }
            }
        }
        asm volatile("s_waitcnt vmcnt(0)" ::: "memory");
        if (lane == 0)
            __hip_atomic_store(flag, call * 8u + (unsigned)(ROUNDS - 1), __ATOMIC_RELAXED, __HIP_MEMORY_SCOPE_SYSTEM);
    } else if (wave == 1) {
        const int xcd = b & 7, idx = b >> 3;
        const int ng = 4 * xcd + (idx & 3);
        const int mt = idx >> 2;
        __syncthreads();
        const unsigned call = __builtin_amdgcn_readfirstlane(s_call);
        const int us = lane >> 3, uj = lane & 7;
        gu32 pfw = (gu32)(ws_me + OFF_FLAG + (size_t)(us * 32 + ng) * 128);
        gu32 pfx = (gu32)(ws_me + OFF_FLAG + (size_t)(us * 32 + 4 * mt + (uj >> 2)) * 128);
        const unsigned seqw = call * 8u + (unsigned)uj, seqx = call * 8u;
        unsigned long long xr = 0ull, m0 = 0ull;
        const unsigned long long t0 = __builtin_amdgcn_s_memrealtime();
        unsigned spin = 0;
        while (m0 != ~0ull) {
            if (xr != ~0ull) {
                unsigned vx0 = __hip_atomic_load(pfx, __ATOMIC_RELAXED, __HIP_MEMORY_SCOPE_SYSTEM);
                unsigned vx1 = __hip_atomic_load(pfx + 64, __ATOMIC_RELAXED, __HIP_MEMORY_SCOPE_SYSTEM);
                xr = __ballot((int)(vx0 - seqx) >= 0 && (int)(vx1 - seqx) >= 0);
            }
            unsigned vw = __hip_atomic_load(pfw, __ATOMIC_RELAXED, __HIP_MEMORY_SCOPE_SYSTEM);
            const unsigned long long n0 = __ballot((int)(vw - seqw) >= 0) & xr;
            if (n0 != m0) {
                m0 = n0;
                if (lane == 0) __hip_atomic_store(&s_mask, m0, __ATOMIC_RELAXED, __HIP_MEMORY_SCOPE_WORKGROUP);
            } else if ((++spin & 63u) == 0u) {
                if (__hip_atomic_load(tmo, __ATOMIC_RELAXED, __HIP_MEMORY_SCOPE_AGENT) == call ||
                    __builtin_amdgcn_s_memrealtime() - t0 > DEADLINE_TICKS) {
                    if (lane == 0) {
                        __hip_atomic_store(tmo, call, __ATOMIC_RELAXED, __HIP_MEMORY_SCOPE_AGENT);
                        __hip_atomic_store(&s_abort, 1u, __ATOMIC_RELAXED, __HIP_MEMORY_SCOPE_WORKGROUP);
                    }
                    break;
                }
            }
            if (__builtin_popcountll(m0) < 48) __builtin_amdgcn_s_sleep(64);
            else __builtin_amdgcn_s_sleep(2);
        }
    } else {
        const int cw = wave - 2;
        const int xcd = b & 7, idx = b >> 3;
        const int ng = 4 * xcd + (idx & 3);
        const int mt = idx >> 2;
        __syncthreads();
        const unsigned call = __builtin_amdgcn_readfirstlane(s_call);
        char* ws_u = uniform_ptr(ws_me);
        auto rl = __builtin_amdgcn_make_buffer_rsrc(ws_u, 0, (int)WS_NEEDED, 0x00020000);
        const unsigned buf = (unsigned)(OFF_DATA + (call & 1u) * BUF_BYTES);
        const unsigned wb = buf + (unsigned)ng * 8192 + lane * 16;
        const unsigned xb = buf + (unsigned)W_BYTES + (unsigned)(2 * mt) * 2048 + lane * 16;

        v16f acc[2][4];
        #pragma unroll
        for (int m = 0; m < 2; ++m)
            #pragma unroll
            for (int q = 0; q < 4; ++q)
                #pragma unroll
                for (int r = 0; r < 16; ++r) acc[m][q][r] = 0.f;

        bool ok = true;
        unsigned long long done = cw ? 0x00ff00ff00ff00ffull : 0xff00ff00ff00ff00ull;
        unsigned tmo_seen = 0;
        while (done != ~0ull) {
            unsigned long long msk = __hip_atomic_load(&s_mask, __ATOMIC_RELAXED, __HIP_MEMORY_SCOPE_WORKGROUP);
            unsigned long long pend;
            {
                unsigned lo = __builtin_amdgcn_readfirstlane((unsigned)msk);
                unsigned hi = __builtin_amdgcn_readfirstlane((unsigned)(msk >> 32));
                pend = (((unsigned long long)hi << 32) | lo) & ~done;
            }
            if (pend == 0ull) {
                if (__builtin_amdgcn_readfirstlane(__hip_atomic_load(&s_abort, __ATOMIC_RELAXED, __HIP_MEMORY_SCOPE_WORKGROUP)) != 0u) {
                    ok = false;
                    break;
                }
                __builtin_amdgcn_s_sleep(1);
                continue;
            }
            __builtin_amdgcn_fence(__ATOMIC_ACQUIRE, "wavefront");
            tmo_seen = __hip_atomic_load(tmo, __ATOMIC_RELAXED, __HIP_MEMORY_SCOPE_AGENT);
            v4i fa[BATCH][2][2], fb[BATCH][4][2];
            int cnt = 0;
            #pragma unroll
            for (int i = 0; i < BATCH; ++i) {
                unsigned wp, xp;
                if (pend != 0ull) {
                    const unsigned u = (unsigned)__builtin_ctzll(pend);
                    pend &= pend - 1ull;
                    done |= 1ull << u;
                    wp = wb + u * (32 * 8192);
                    xp = xb + u * (16 * 2048);
                    cnt = i + 1;
                } else {
                    wp = 0x80000000u; xp = 0x80000000u;
                }
                #pragma unroll
                for (int m = 0; m < 2; ++m) {
                    fa[i][m][0] = LOADF(xp + m * 2048); fa[i][m][1] = LOADF(xp + m * 2048 + 1024);
                }
                #pragma unroll
                for (int q = 0; q < 4; ++q) {
                    fb[i][q][0] = LOADF(wp + q * 2048); fb[i][q][1] = LOADF(wp + q * 2048 + 1024);
                }
            }
            __builtin_amdgcn_sched_barrier(0);
            #pragma unroll
            for (int i = 0; i < BATCH; ++i) {
                if (i < cnt) {
                    #pragma unroll
                    for (int q = 0; q < 4; ++q) {
                        v4i b0 = fb[i][q][0], b1 = fb[i][q][1];
                        v8i bb = {b0[0], b0[1], b0[2], b0[3], b1[0], b1[1], b1[2], b1[3]};
                        #pragma unroll
                        for (int m = 0; m < 2; ++m) {
                            v4i a0 = fa[i][m][0], a1 = fa[i][m][1];
                            v8i a = {a0[0], a0[1], a0[2], a0[3], a1[0], a1[1], a1[2], a1[3]};
                            acc[m][q] = __builtin_amdgcn_mfma_scale_f32_32x32x64_f8f6f4(a, bb, acc[m][q], 0, 0, 0, 0x7f7f7f7f, 0, 0x7f7f7f7f);
                        }
                    }
                }
            }
        }
        if (tmo_seen == call) ok = false;
        {
            v4f* dst = (v4f*)(s_red + cw * 16384) + lane;
            #pragma unroll
            for (int q = 0; q < 4; ++q)
                #pragma unroll
                for (int g = 0; g < 4; ++g) {
                    v4f v;
                    if (cw == 0) { v[0] = acc[1][q][4 * g]; v[1] = acc[1][q][4 * g + 1]; v[2] = acc[1][q][4 * g + 2]; v[3] = acc[1][q][4 * g + 3]; }
                    else         { v[0] = acc[0][q][4 * g]; v[1] = acc[0][q][4 * g + 1]; v[2] = acc[0][q][4 * g + 2]; v[3] = acc[0][q][4 * g + 3]; }
                    dst[(q * 4 + g) * 64] = v;
                }
            asm volatile("s_waitcnt lgkmcnt(0)" ::: "memory");
            if (lane == 0) __hip_atomic_store(&s_rdy[cw], ok ? 1u : 2u, __ATOMIC_RELAXED, __HIP_MEMORY_SCOPE_WORKGROUP);
            unsigned peer;
            while ((peer = __builtin_amdgcn_readfirstlane(__hip_atomic_load(&s_rdy[cw ^ 1], __ATOMIC_RELAXED, __HIP_MEMORY_SCOPE_WORKGROUP))) == 0u)
                __builtin_amdgcn_s_sleep(1);
            if (peer != 1u) ok = false;
            asm volatile("" ::: "memory");
        }
        v16f fin[4];
        {
            const v4f* src = (const v4f*)(s_red + (cw ^ 1) * 16384) + lane;
            #pragma unroll
            for (int q = 0; q < 4; ++q) {
                fin[q] = cw ? acc[1][q] : acc[0][q];
                #pragma unroll
                for (int g = 0; g < 4; ++g) {
                    v4f v = src[(q * 4 + g) * 64];
                    fin[q][4 * g] += v[0]; fin[q][4 * g + 1] += v[1]; fin[q][4 * g + 2] += v[2]; fin[q][4 * g + 3] += v[3];
                }
            }
        }
        const float scale = ok ? (sx[0] * sw[0]) : __builtin_nanf("");
        auto ro = __builtin_amdgcn_make_buffer_rsrc(out, 0, MS * GN * 4, 0x00020000);
        const unsigned oo = (unsigned)(((64 * mt + 32 * cw + 4 * h) * GN + 128 * ng + 4 * jj) * 4);
        #pragma unroll
        for (int r = 0; r < 16; ++r) {
            v4f v;
            if (ok) { v[0] = fin[0][r] * scale; v[1] = fin[1][r] * scale; v[2] = fin[2][r] * scale; v[3] = fin[3][r] * scale; }
            else { v[0] = v[1] = v[2] = v[3] = scale; }
            __builtin_amdgcn_raw_buffer_store_b128(__builtin_bit_cast(v4i, v), ro, oo + (unsigned)(((r & 3) + 8 * (r >> 2)) * GN * 4), 0, 0);
        }
    }
}

extern "C" void kernel_launch(int rank, int nranks, ncclComm_t comm,
                              hipStream_t stream,
                              void* const* d_in, const int* in_sizes, int n_in,
                              void* d_out, int out_size,
                              const int* in_dtypes, int out_dtype,
                              void* const* d_out_all,
                              void* const* d_ws_all, size_t ws_bytes) {
    if (nranks != NR || ws_bytes < WS_NEEDED || (unsigned)rank >= (unsigned)NR) {
        fprintf(stderr, "unsupported configuration\n");
        return;
    }
    gemm_ag_kernel<<<NWG, 256, 0, stream>>>((const uint8_t*)d_in[0], (const uint8_t*)d_in[1],
                                            (const float*)d_in[2], (const float*)d_in[3],
                                            (float*)d_out, d_ws_all, rank);
    HIP_CHECK(hipGetLastError());
}
```
